```python
import numpy as np

kernel_name = "gpu_ag_gemm_m8192_k8192_n8192_bf16_none"


ABSMAX_RTOL = 0.008

COMPUTE_DTYPE = "bfloat16"
EPILOGUE = "none"
IN_QUANT = None
OUT_QUANT = None
IN_DTYPES = {"x": "bfloat16", "w_mat": "bfloat16"}
WORLD_SIZE = 8

def _to_bf16(x):
    u = np.ascontiguousarray(x, dtype=np.float32).view(np.uint32)
    return ((u + 0x7FFF + ((u >> 16) & 1)) & 0xFFFF0000).view(np.float32)

def setup_inputs():

    x = _to_bf16(np.random.randn(8192, 8192).astype(np.float32))
    w_mat = _to_bf16((np.random.randn(8192, 8192) / np.sqrt(8192)).astype(np.float32))
    return {"x": x, "w_mat": w_mat}

def _epilogue(y):
    return y

def reference(x, w_mat):
    y = (x @ w_mat).astype(np.float32)
    return _epilogue(y)
```

```cpp
#include <hip/hip_runtime.h>
#include <rccl/rccl.h>
#include <cstdio>
#include <cstdint>

#define HIP_CHECK(expr) do { hipError_t _e = (expr); if (_e != hipSuccess) { \
    fprintf(stderr, "HIP error %s at %s:%d\n", hipGetErrorString(_e), __FILE__, __LINE__); \
    return; } } while (0)

namespace {

#define LAS __attribute__((address_space(3)))
typedef unsigned short bf16_t;
typedef short bf16x8 __attribute__((ext_vector_type(8)));
typedef float f32x4 __attribute__((ext_vector_type(4)));
typedef unsigned u32x4 __attribute__((ext_vector_type(4)));

constexpr int NR = 8;
constexpr int MSH = 1024;
constexpr int KD = 8192;
constexpr int NSH = 1024;
constexpr int MALL = NR * MSH;
constexpr int KC = 256;
constexpr int NCH = KD / KC;
constexpr int TPC = KC / 64;
constexpr int NCOPY = 64;
constexpr int ROWS_PER_CWG = MSH / NCOPY;
static_assert(ROWS_PER_CWG * KC * 2 == 512 * 16, "one 16-byte piece per thread per chunk");
constexpr size_t SLOT = (size_t)MSH * KD * 2;
constexpr size_t GBUF = (size_t)NR * SLOT;
constexpr size_t CTRL = 2 * GBUF;
constexpr size_t FLAG_BYTES = (size_t)NR * NCH * NCOPY * 4;
constexpr size_t CTR_COPY = CTRL + FLAG_BYTES;
constexpr size_t CTR_GEMM = CTR_COPY + 128;
constexpr size_t TMO = CTR_GEMM + 128;
constexpr size_t WS_NEED = TMO + 128;
constexpr unsigned long long DEADLINE_TICKS = 40000000ull;

constexpr int BM = 256, BK = 64, HALF = 128, HTB = HALF * BK * 2, STAGE_BYTES = 8 * HTB;
constexpr int LDS_BYTES = STAGE_BYTES + 16;
constexpr int NT = KD / BK;

struct Params {
    const bf16_t* x;
    const bf16_t* w;
    bf16_t* out;
    char* ws[NR];
    int rank;
    int phase;
};

__host__ __device__ __forceinline__ int lds_byte(int r, int c) { const int st = (r >> 4) * 2 + (c >> 5), rr = r & 15, cc = c & 31, ob = rr * 64 + cc * 2; return st * 1024 + (ob ^ (((ob >> 9) & 1) << 5)); }
__host__ __device__ __forceinline__ void stage_rc(int b, int& R, int& C) { const int st = b / 1024, sb = b % 1024, swz = sb ^ (((sb >> 9) & 1) << 5); R = (st >> 1) * 16 + swz / 64; C = (st & 1) * 32 + (swz % 64) / 2; }
__host__ __device__ __forceinline__ int perm32(int rho) { const int n = rho >> 4, i = rho & 15; return 8 * (i >> 2) + 4 * n + (i & 3); }
__device__ __forceinline__ unsigned cvt_pk_bf16(float lo, float hi) { unsigned r; asm volatile("v_cvt_pk_bf16_f32 %0, %1, %2" : "=v"(r) : "v"(lo), "v"(hi)); return r; }

__device__ __forceinline__ unsigned derive_call(unsigned* ctr, LAS unsigned* slot) {
    if (threadIdx.x == 0) {
        const unsigned t = __hip_atomic_fetch_add(ctr, 1u, __ATOMIC_RELAXED, __HIP_MEMORY_SCOPE_AGENT);
        *slot = t / gridDim.x + 1u;
    }
    __syncthreads();
    const unsigned c = __builtin_amdgcn_readfirstlane(*slot);
    __syncthreads();
    return c;
}

__device__ __forceinline__ void copy_phase(const Params& p, LAS unsigned char* lds) {
    char* myws = p.ws[p.rank];
    const unsigned call = derive_call((unsigned*)(myws + CTR_COPY), (LAS unsigned*)lds);
    const int tid = threadIdx.x, wg = blockIdx.x;
    const size_t par = (size_t)(call & 1u) * GBUF;
    const int r = wg * ROWS_PER_CWG + (tid >> 5), cb = (tid & 31) * 16;
    const char* src = (const char*)p.x + (size_t)r * (KD * 2) + cb;
    const size_t dst_off = par + (size_t)p.rank * SLOT + (size_t)r * (KC * 2) + cb;
    const size_t flag_off = CTRL + ((size_t)p.rank * NCH * NCOPY + wg) * 4;
    unsigned* myflag = (unsigned*)(p.ws[(p.rank + 1 + (tid % (NR - 1))) & (NR - 1)] + flag_off);
    char* dsts[NR - 1];
#pragma unroll
    for (int q = 1; q < NR; ++q) dsts[q - 1] = p.ws[(p.rank + q) & (NR - 1)] + dst_off;
    u32x4 v, vn;
    asm volatile("global_load_dwordx4 %0, %1, off nt" : "=v"(v) : "v"(src) : "memory");
    asm volatile("s_waitcnt vmcnt(0)" : "+v"(v) :: "memory");
    for (int c = 0; c < NCH; ++c) {
        const char* nsrc = src + (size_t)((c + 1 < NCH) ? c + 1 : c) * (KC * 2);
        asm volatile("global_load_dwordx4 %0, %1, off nt" : "=v"(vn) : "v"(nsrc) : "memory");
#pragma unroll
        for (int q = 0; q < NR - 1; ++q) {
            asm volatile("global_store_dwordx4 %0, %1, off sc0 sc1" :: "v"(dsts[q] + (size_t)c * (MSH * KC * 2)), "v"(v) : "memory");
        }
        asm volatile("s_waitcnt vmcnt(8)" ::: "memory");
        if (c > 0) {
            __builtin_amdgcn_s_barrier();
            if (tid < NR - 1)
                __hip_atomic_store(myflag + (size_t)(c - 1) * NCOPY, call, __ATOMIC_RELAXED, __HIP_MEMORY_SCOPE_SYSTEM);
        }
        asm volatile("s_waitcnt vmcnt(7)" : "+v"(vn) :: "memory");
        v = vn;
    }
    asm volatile("s_waitcnt vmcnt(0)" ::: "memory");
    __syncthreads();
    if (tid < NR - 1)
        __hip_atomic_store(myflag + (size_t)(NCH - 1) * NCOPY, call, __ATOMIC_RELAXED, __HIP_MEMORY_SCOPE_SYSTEM);
}

__device__ __forceinline__ void transpose_phase(const Params& p, LAS unsigned char* lds) {
    bf16_t* Bt = (bf16_t*)(p.ws[p.rank] + (size_t)p.rank * SLOT);
    LAS bf16_t* t = (LAS bf16_t*)lds;
    constexpr int PITCH = 66;
    const int tid = threadIdx.x, a = tid >> 3, b = (tid & 7) * 8;
    constexpr int NTILE = (KD / 64) * (NSH / 64);
    for (int tile = blockIdx.x; tile < NTILE; tile += gridDim.x) {
        const int k0 = (tile >> 4) * 64, n0 = (tile & 15) * 64;
        const bf16x8 v = *(const bf16x8*)(p.w + (size_t)(k0 + a) * NSH + n0 + b);
#pragma unroll
        for (int j = 0; j < 8; ++j) t[a * PITCH + b + j] = (bf16_t)v[j];
        __syncthreads();
        bf16x8 o;
#pragma unroll
        for (int j = 0; j < 8; ++j) o[j] = (short)t[(b + j) * PITCH + a];
        *(bf16x8*)(Bt + (size_t)(n0 + a) * KD + k0 + b) = o;
        __syncthreads();
    }
}

__device__ __forceinline__ void wait_chunk(const unsigned* flags, unsigned call, unsigned* tmo) {
    if (threadIdx.x < 64) {
        const unsigned long long t0 = __builtin_amdgcn_s_memrealtime();
        unsigned polls = 0;
        for (;;) {
            const unsigned v = __hip_atomic_load(flags + threadIdx.x, __ATOMIC_RELAXED, __HIP_MEMORY_SCOPE_SYSTEM);
            if (__all(v >= call)) break;
            if (((++polls & 63u) == 0u && __hip_atomic_load(tmo, __ATOMIC_RELAXED, __HIP_MEMORY_SCOPE_AGENT) == call)
                || __builtin_amdgcn_s_memrealtime() - t0 > DEADLINE_TICKS) {
                __hip_atomic_store(tmo, call, __ATOMIC_RELAXED, __HIP_MEMORY_SCOPE_AGENT);
                break;
            }
            __builtin_amdgcn_s_sleep(1);
        }
        __builtin_amdgcn_fence(__ATOMIC_ACQUIRE, "");
        asm volatile("s_waitcnt vmcnt(0)" ::: "memory");
    }
    asm volatile("" ::: "memory"); __builtin_amdgcn_s_barrier(); asm volatile("" ::: "memory");
}

__device__ __forceinline__ void gemm_phase(const Params& p, LAS unsigned char* lds) {
    char* myws = p.ws[p.rank];
    const unsigned call = derive_call((unsigned*)(myws + CTR_GEMM), (LAS unsigned*)(lds + STAGE_BYTES));
    unsigned* tmo = (unsigned*)(myws + TMO);
    const int tid = threadIdx.x, wid = __builtin_amdgcn_readfirstlane(tid >> 6), lane = tid & 63, wr = wid >> 2, wc = wid & 3, fr = lane & 15, fq = lane >> 4;
    const int b = blockIdx.x, src = b & 7, idx = b >> 3, pml = idx & 3, pn = idx >> 2, pm = src * 4 + pml;
    const bool remote = (src != p.rank);
    const unsigned rstride = remote ? (unsigned)(KC * 2) : (unsigned)(KD * 2);
    const size_t cstride = remote ? (size_t)MSH * KC * 2 : (size_t)KC * 2;
    const char* baseA = remote ? (const char*)(myws + (size_t)(call & 1u) * GBUF + (size_t)src * SLOT) : (const char*)p.x;
    const unsigned* flags = (const unsigned*)(myws + CTRL) + (size_t)src * NCH * NCOPY;
    const bf16_t* Bt = (const bf16_t*)(myws + (size_t)p.rank * SLOT);

    unsigned voffA[2], voffB[2];
#pragma unroll
    for (int i = 0; i < 2; ++i) { int R, C; stage_rc(tid * 16 + i * 8192, R, C); const int Rb = (R & ~31) + perm32(R & 31);
        voffA[i] = (unsigned)R * rstride + (unsigned)C * 2u; voffB[i] = (unsigned)(Rb * KD + C) * 2u; }
    const size_t kstep = (size_t)(BK * 2);
    const size_t hstepA = (size_t)HALF * rstride, hstepB = (size_t)HALF * KD * 2;
    const unsigned ldsw = (unsigned)wid * 1024u;
    const int aoff = lds_byte(wr * 64 + fr, fq * 8), boff = lds_byte(wc * 32 + fr, fq * 8);
#define G_SA(b, h) (((b) * 2 + (h)) * HTB)
#define G_SB(b, h) ((4 + (b) * 2 + (h)) * HTB)
#define G_STAGE(bufoff, gbase, voff) do { _Pragma("unroll") for (int _i = 0; _i < 2; ++_i) \
        __builtin_amdgcn_global_load_lds((const unsigned*)((const char*)(gbase) + (voff)[_i]), (LAS unsigned*)(lds + (bufoff) + ldsw + _i * 8192), 16, 0, 0); } while (0)
#define G_LDA(dst, b, h) do { _Pragma("unroll") for (int m = 0; m < 4; ++m) _Pragma("unroll") for (int k = 0; k < 2; ++k) dst[m][k] = *(const LAS bf16x8*)(lds + G_SA(b, h) + aoff + m * 2048 + k * 1024); } while (0)
#define G_LDB(dst, b, h) do { _Pragma("unroll") for (int n = 0; n < 2; ++n) _Pragma("unroll") for (int k = 0; k < 2; ++k) dst[n][k] = *(const LAS bf16x8*)(lds + G_SB(b, h) + boff + n * 2048 + k * 1024); } while (0)
#define G_MMA(ai, bj, At, Bt_) do { __builtin_amdgcn_s_setprio(1); _Pragma("unroll") for (int m = 0; m < 4; ++m) _Pragma("unroll") for (int n = 0; n < 2; ++n) _Pragma("unroll") for (int k = 0; k < 2; ++k) \
        acc[ai][bj][m][n] = __builtin_amdgcn_mfma_f32_16x16x32_bf16(Bt_[n][k], At[m][k], acc[ai][bj][m][n], 0, 0, 0); __builtin_amdgcn_s_setprio(0); } while (0)
#define G_WAIT_V(n) asm volatile("s_waitcnt vmcnt(" #n ")" ::: "memory")
#define G_WAIT_L(n) asm volatile("s_waitcnt lgkmcnt(" #n ")" ::: "memory")
#define G_BAR __builtin_amdgcn_s_barrier()
#define G_SCHED __builtin_amdgcn_sched_barrier(0)
    f32x4 acc[2][2][4][2];
#pragma unroll
    for (int a = 0; a < 2; ++a)
#pragma unroll
        for (int bb = 0; bb < 2; ++bb)
#pragma unroll
            for (int m = 0; m < 4; ++m)
#pragma unroll
                for (int n = 0; n < 2; ++n) acc[a][bb][m][n] = (f32x4){0.f, 0.f, 0.f, 0.f};
    bf16x8 At[4][2], B0[2][2], B1[2][2];
    const char* cA = baseA + (size_t)pml * BM * rstride;
    const char* cB = (const char*)Bt + (size_t)pn * BM * KD * 2;
    if (remote) wait_chunk(flags, call, tmo);
    G_STAGE(G_SB(0, 0), cB, voffB); G_STAGE(G_SA(0, 0), cA, voffA); G_STAGE(G_SB(0, 1), cB + hstepB, voffB); G_STAGE(G_SA(0, 1), cA + hstepA, voffA);
    if (wr == 1) G_BAR;
    G_WAIT_V(4); G_BAR;
    G_STAGE(G_SB(1, 0), cB + kstep, voffB); G_STAGE(G_SA(1, 0), cA + kstep, voffA); G_STAGE(G_SB(1, 1), cB + hstepB + kstep, voffB);
    G_WAIT_V(6); G_BAR;
    for (int t = 0; t < NT; t += 2) {
        const bool last = (t == NT - 2);
        const char* a0 = cA + (size_t)(t / TPC) * cstride + (size_t)(t % TPC) * kstep;
        const char* a1 = a0 + kstep;
        const char* a2 = last ? cA : cA + (size_t)((t + 2) / TPC) * cstride + (size_t)((t + 2) % TPC) * kstep;
        const char* b2 = last ? cB : cB + (size_t)(t + 2) * kstep;
        const char* a3 = a2 + kstep; const char* b3 = b2 + kstep;
        if (remote && !last && ((t + 2) % TPC) == 0) wait_chunk(flags + ((t + 2) / TPC) * NCOPY, call, tmo);
        G_LDB(B0, 0, 0); G_SCHED; G_LDA(At, 0, 0); G_STAGE(G_SA(1, 1), a1 + hstepA, voffA);
        G_WAIT_L(8); G_BAR; G_WAIT_L(0); G_MMA(0, 0, At, B0); G_BAR; G_SCHED;
        G_LDB(B1, 0, 1); G_STAGE(G_SB(0, 0), b2, voffB);
        G_BAR; G_WAIT_L(0); G_MMA(0, 1, At, B1); G_BAR;
        G_LDA(At, 0, 1); G_STAGE(G_SA(0, 0), a2, voffA);
        G_BAR; G_WAIT_L(0); G_MMA(1, 0, At, B0); G_BAR; G_SCHED;
        G_STAGE(G_SB(0, 1), b2 + hstepB, voffB);
        G_WAIT_V(6); G_BAR; G_MMA(1, 1, At, B1); G_BAR;
        G_LDB(B0, 1, 0); G_SCHED; G_LDA(At, 1, 0); G_STAGE(G_SA(0, 1), a2 + hstepA, voffA);
        G_WAIT_L(8); G_BAR; G_WAIT_L(0); G_MMA(0, 0, At, B0); G_BAR; G_SCHED;
        G_LDB(B1, 1, 1); G_STAGE(G_SB(1, 0), b3, voffB);
        G_BAR; G_WAIT_L(0); G_MMA(0, 1, At, B1); G_BAR;
        G_LDA(At, 1, 1); G_STAGE(G_SA(1, 0), a3, voffA);
        G_BAR; G_WAIT_L(0); G_MMA(1, 0, At, B0); G_BAR; G_SCHED;
        G_STAGE(G_SB(1, 1), b3 + hstepB, voffB);
        G_WAIT_V(6); G_BAR; G_MMA(1, 1, At, B1); G_BAR;
    }
    {
        const bool bad = (__hip_atomic_load(tmo, __ATOMIC_RELAXED, __HIP_MEMORY_SCOPE_AGENT) == call);
        const int row0 = pm * BM + wr * 64 + fr, col0 = pn * BM + wc * 32 + 8 * fq;
#pragma unroll
        for (int ai = 0; ai < 2; ++ai)
#pragma unroll
            for (int m = 0; m < 4; ++m) {
                bf16_t* rowp = p.out + (size_t)(row0 + ai * HALF + m * 16) * NSH + col0;
#pragma unroll
                for (int bj = 0; bj < 2; ++bj) {
                    const f32x4 v0 = acc[ai][bj][m][0], v1 = acc[ai][bj][m][1];
                    u32x4 w; w.x = cvt_pk_bf16(v0[0], v0[1]); w.y = cvt_pk_bf16(v0[2], v0[3]); w.z = cvt_pk_bf16(v1[0], v1[1]); w.w = cvt_pk_bf16(v1[2], v1[3]);
                    if (bad) { w.x = 0x7fc07fc0u; w.y = 0x7fc07fc0u; w.z = 0x7fc07fc0u; w.w = 0x7fc07fc0u; }
                    *(u32x4*)(rowp + bj * HALF) = w;
                }
            }
    }
    G_WAIT_V(0);
    if (wr == 0) G_BAR;
    G_BAR;
}

__global__ __launch_bounds__(512, 2) void ag_gemm_kernel(Params p) {
    extern __shared__ __attribute__((aligned(16))) unsigned char shm[];
    LAS unsigned char* lds = (LAS unsigned char*)shm;
    if (p.phase == 0) copy_phase(p, lds);
    else if (p.phase == 1) transpose_phase(p, lds);
    else gemm_phase(p, lds);
}

}

extern "C" void kernel_launch(int rank, int nranks, ncclComm_t comm,
                              hipStream_t stream,
                              void* const* d_in, const int* in_sizes, int n_in,
                              void* d_out, int out_size,
                              const int* in_dtypes, int out_dtype,
                              void* const* d_out_all,
                              void* const* d_ws_all, size_t ws_bytes) {
    static hipStream_t side[NR] = {};
    static hipEvent_t ev_fork[NR] = {}, ev_join[NR] = {};
    if (nranks != NR || rank < 0 || rank >= NR || ws_bytes < WS_NEED) { fprintf(stderr, "unsupported configuration\n"); return; }
    if (!side[rank]) {
        HIP_CHECK(hipStreamCreateWithFlags(&side[rank], hipStreamNonBlocking));
        HIP_CHECK(hipEventCreateWithFlags(&ev_fork[rank], hipEventDisableTiming));
        HIP_CHECK(hipEventCreateWithFlags(&ev_join[rank], hipEventDisableTiming));
        HIP_CHECK(hipFuncSetAttribute((const void*)ag_gemm_kernel, hipFuncAttributeMaxDynamicSharedMemorySize, LDS_BYTES));
    }
    Params p;
    p.x = (const bf16_t*)d_in[0];
    p.w = (const bf16_t*)d_in[1];
    p.out = (bf16_t*)d_out;
    for (int i = 0; i < NR; ++i) p.ws[i] = (char*)d_ws_all[i];
    p.rank = rank;

    HIP_CHECK(hipEventRecord(ev_fork[rank], stream));
    HIP_CHECK(hipStreamWaitEvent(side[rank], ev_fork[rank], 0));
    p.phase = 0;
    ag_gemm_kernel<<<NCOPY, 512, 16, side[rank]>>>(p);
    HIP_CHECK(hipEventRecord(ev_join[rank], side[rank]));
    p.phase = 1;
    ag_gemm_kernel<<<256, 512, 16384, stream>>>(p);
    p.phase = 2;
    ag_gemm_kernel<<<128, 512, LDS_BYTES, stream>>>(p);
    HIP_CHECK(hipStreamWaitEvent(stream, ev_join[rank], 0));
    HIP_CHECK(hipGetLastError());
}
```

```cpp
#include <hip/hip_runtime.h>
#include <rccl/rccl.h>
#include <cstdio>
#include <cstdint>

#define HIP_CHECK(expr) do { hipError_t _e = (expr); if (_e != hipSuccess) { \
    fprintf(stderr, "HIP error %s at %s:%d\n", hipGetErrorString(_e), __FILE__, __LINE__); \
    return; } } while (0)

namespace {

#define LAS __attribute__((address_space(3)))
typedef unsigned short bf16_t;
typedef short bf16x8 __attribute__((ext_vector_type(8)));
typedef short s16x4 __attribute__((ext_vector_type(4)));
typedef unsigned u32x2 __attribute__((ext_vector_type(2)));
typedef float f32x4 __attribute__((ext_vector_type(4)));
typedef unsigned u32x4 __attribute__((ext_vector_type(4)));

constexpr int NR = 8;
constexpr int MSH = 1024;
constexpr int KD = 8192;
constexpr int NSH = 1024;
constexpr int MALL = NR * MSH;
constexpr int KC = 256;
constexpr int NCH = KD / KC;
constexpr int TPC = KC / 64;
constexpr int NCOPY = 16;
constexpr int PPT = MSH * KC * 2 / (NCOPY * 512 * 16);
constexpr int TPR = KC * 2 / 16;
constexpr int RPP = 512 / TPR;
static_assert(PPT * NCOPY * 512 * 16 == MSH * KC * 2 && PPT * 7 <= 56, "copy geometry");
constexpr size_t SLOT = (size_t)MSH * KD * 2;
constexpr size_t GBUF = (size_t)NR * SLOT;
constexpr size_t CTRL = 2 * GBUF;
constexpr size_t FLAG_BYTES = (size_t)NR * NCH * NCOPY * 4;
constexpr size_t CTR = CTRL + FLAG_BYTES;
constexpr size_t TMO = CTR + 128;
constexpr size_t WS_NEED = TMO + 128;
#define STAMP(ws, slot, i) do {} while (0)
constexpr unsigned long long DEADLINE_TICKS = 40000000ull;

constexpr int BM = 256, BK = 64, HALF = 128, HTB = HALF * BK * 2, STAGE_BYTES = 8 * HTB;
constexpr int LDS_BYTES = STAGE_BYTES + 16;
constexpr int NT = KD / BK;

struct Params {
    const bf16_t* x;
    const bf16_t* w;
    bf16_t* out;
    char* ws[NR];
    int rank;
};

__host__ __device__ __forceinline__ int lds_byte(int r, int c) { const int st = (r >> 4) * 2 + (c >> 5), rr = r & 15, cc = c & 31, ob = rr * 64 + cc * 2; return st * 1024 + (ob ^ (((ob >> 9) & 1) << 5)); }
__host__ __device__ __forceinline__ void stage_rc(int b, int& R, int& C) { const int st = b / 1024, sb = b % 1024, swz = sb ^ (((sb >> 9) & 1) << 5); R = (st >> 1) * 16 + swz / 64; C = (st & 1) * 32 + (swz % 64) / 2; }
__host__ __device__ __forceinline__ int perm32(int rho) { const int n = rho >> 4, i = rho & 15; return 8 * (i >> 2) + 4 * n + (i & 3); }
__device__ __forceinline__ unsigned cvt_pk_bf16(float lo, float hi) { unsigned r; asm volatile("v_cvt_pk_bf16_f32 %0, %1, %2" : "=v"(r) : "v"(lo), "v"(hi)); return r; }

__device__ __forceinline__ unsigned derive_call(unsigned* ctr, LAS unsigned* slot) {
    if (threadIdx.x == 0) {
        const unsigned t = __hip_atomic_fetch_add(ctr, 1u, __ATOMIC_RELAXED, __HIP_MEMORY_SCOPE_AGENT);
        *slot = t / gridDim.x + 1u;
    }
    __syncthreads();
    const unsigned c = __builtin_amdgcn_readfirstlane(*slot);
    __syncthreads();
    return c;
}

__device__ __forceinline__ void copy_phase(const Params& p, const unsigned call) {
    char* myws = p.ws[p.rank];
    const int tid = threadIdx.x, wg = blockIdx.x;
    const size_t par = (size_t)(call & 1u) * GBUF;
    const int r0 = wg * (RPP * PPT) + tid / TPR, cb = (tid % TPR) * 16;
    const char* src = (const char*)p.x + (size_t)r0 * (KD * 2) + cb;
    const size_t dst_off = par + (size_t)p.rank * SLOT + (size_t)r0 * (KC * 2) + cb;
    const size_t flag_off = CTRL + ((size_t)p.rank * NCH * NCOPY + wg) * 4;
    unsigned* myflag = (unsigned*)(p.ws[(p.rank + 1 + (tid % (NR - 1))) & (NR - 1)] + flag_off);
    char* dsts[NR - 1];
#pragma unroll
    for (int q = 1; q < NR; ++q) dsts[q - 1] = p.ws[(p.rank + q) & (NR - 1)] + dst_off;
    STAMP(myws, 128 + wg, 0);
    for (int c = 0; c < NCH; ++c) {
        u32x4 v[PPT];
#pragma unroll
        for (int j = 0; j < PPT; ++j)
            asm volatile("global_load_dwordx4 %0, %1, off nt" : "=v"(v[j]) : "v"(src + (size_t)j * (RPP * KD * 2) + (size_t)c * (KC * 2)) : "memory");
        asm volatile("s_waitcnt vmcnt(0)" ::: "memory");
#pragma unroll
        for (int j = 0; j < PPT; ++j) asm volatile("" : "+v"(v[j]));
        if (c > 0) {
            __builtin_amdgcn_s_barrier();
            if (tid < NR - 1)
                __hip_atomic_store(myflag + (size_t)(c - 1) * NCOPY, call, __ATOMIC_RELAXED, __HIP_MEMORY_SCOPE_SYSTEM);
            STAMP(myws, 128 + wg, c);
        }
#pragma unroll
        for (int j = 0; j < PPT; ++j)
#pragma unroll
            for (int q = 0; q < NR - 1; ++q)
                asm volatile("global_store_dwordx4 %0, %1, off sc0 sc1\n\ts_nop 1" :: "v"(dsts[q] + (size_t)j * (RPP * KC * 2) + (size_t)c * (MSH * KC * 2)), "v"(v[j]) : "memory");
    }
    asm volatile("s_waitcnt vmcnt(0)" ::: "memory");
    __syncthreads();
    if (tid < NR - 1)
        __hip_atomic_store(myflag + (size_t)(NCH - 1) * NCOPY, call, __ATOMIC_RELAXED, __HIP_MEMORY_SCOPE_SYSTEM);
    STAMP(myws, 128 + wg, NCH);
}

__device__ __forceinline__ void wait_chunk(const unsigned* flags, unsigned call, unsigned* tmo) {
    if (threadIdx.x < 64) {
        const unsigned long long t0 = __builtin_amdgcn_s_memrealtime();
        unsigned polls = 0;
        for (;;) {
            const unsigned v = __hip_atomic_load(flags + (threadIdx.x % NCOPY), __ATOMIC_RELAXED, __HIP_MEMORY_SCOPE_SYSTEM);
            if (__all(v >= call)) break;
            if (((++polls & 63u) == 0u && __hip_atomic_load(tmo, __ATOMIC_RELAXED, __HIP_MEMORY_SCOPE_AGENT) == call)
                || __builtin_amdgcn_s_memrealtime() - t0 > DEADLINE_TICKS) {
                __hip_atomic_store(tmo, call, __ATOMIC_RELAXED, __HIP_MEMORY_SCOPE_AGENT);
                break;
            }
            __builtin_amdgcn_s_sleep(1);
        }
        __builtin_amdgcn_fence(__ATOMIC_ACQUIRE, "");
        asm volatile("s_waitcnt vmcnt(0)" ::: "memory");
    }
    asm volatile("" ::: "memory"); __builtin_amdgcn_s_barrier(); asm volatile("" ::: "memory");
}

__device__ __forceinline__ void gemm_phase(const Params& p, LAS unsigned char* lds, const unsigned call) {
    char* myws = p.ws[p.rank];
    unsigned* tmo = (unsigned*)(myws + TMO);
    const int tid = threadIdx.x, wid = __builtin_amdgcn_readfirstlane(tid >> 6), lane = tid & 63, wr = wid >> 2, wc = wid & 3, fr = lane & 15, fq = lane >> 4;
    const int b = blockIdx.x - NCOPY, src = b & 7, idx = b >> 3, pml = idx & 3, pn = idx >> 2, pm = src * 4 + pml;
    const bool remote = (src != p.rank);
    const unsigned rstride = remote ? (unsigned)(KC * 2) : (unsigned)(KD * 2);
    const size_t cstride = remote ? (size_t)MSH * KC * 2 : (size_t)KC * 2;
    const char* baseA = remote ? (const char*)(myws + (size_t)(call & 1u) * GBUF + (size_t)src * SLOT) : (const char*)p.x;
    const unsigned* flags = (const unsigned*)(myws + CTRL) + (size_t)src * NCH * NCOPY;

    unsigned voffA[2], voffB[2];
#pragma unroll
    for (int i = 0; i < 2; ++i) { int R, C; stage_rc(tid * 16 + i * 8192, R, C);
        voffA[i] = (unsigned)R * rstride + (unsigned)C * 2u;
        const int bb = tid * 16 + i * 8192, row = bb >> 8, ch = ((bb & 255) >> 4) ^ (((row & 3) << 2) | ((row >> 2) & 3));
        voffB[i] = (unsigned)row * (unsigned)(NSH * 2) + (unsigned)ch * 16u; }
    const size_t kstep = (size_t)(BK * 2), kstepB = (size_t)BK * NSH * 2;
    const size_t hstepA = (size_t)HALF * rstride, hstepB = (size_t)HALF * 2;
    const unsigned ldsw = (unsigned)wid * 1024u;
    const int aoff = lds_byte(wr * 64 + fr, fq * 8);
#define G_SA(b, h) (((b) * 2 + (h)) * HTB)
#define G_SB(b, h) ((4 + (b) * 2 + (h)) * HTB)
    int boff[2][2];
#pragma unroll
    for (int n = 0; n < 2; ++n)
#pragma unroll
        for (int t = 0; t < 2; ++t) { const int q = fr >> 2, pp = fr & 3, row = 8 * fq + 4 * t + q;
            boff[n][t] = 256 * row + 16 * ((4 * wc + 2 * n + (pp >> 1)) ^ (((row & 3) << 2) | ((row >> 2) & 3))) + 8 * (pp & 1) + (int)(unsigned)(size_t)lds + G_SB(0, 0); }
#define G_STAGE(bufoff, gbase, voff) do { _Pragma("unroll") for (int _i = 0; _i < 2; ++_i) \
        __builtin_amdgcn_global_load_lds((const unsigned*)((const char*)(gbase) + (voff)[_i]), (LAS unsigned*)(lds + (bufoff) + ldsw + _i * 8192), 16, 0, 0); } while (0)
#define G_LDA(dst, b, h) do { _Pragma("unroll") for (int m = 0; m < 4; ++m) _Pragma("unroll") for (int k = 0; k < 2; ++k) dst[m][k] = *(const LAS bf16x8*)(lds + G_SA(b, h) + aoff + m * 2048 + k * 1024); } while (0)
#define G_TR(dst2, addr, OFF) asm volatile("ds_read_b64_tr_b16 %0, %1 offset:" #OFF : "=v"(dst2) : "v"(addr) : "memory")
#define G_LDB_(dst, O0, O1) do { _Pragma("unroll") for (int n = 0; n < 2; ++n) { u32x2 l0_, h0_, l1_, h1_; \
        G_TR(l0_, boff[n][0], O0); G_TR(h0_, boff[n][1], O0); G_TR(l1_, boff[n][0], O1); G_TR(h1_, boff[n][1], O1); \
        dst[n][0] = __builtin_bit_cast(bf16x8, __builtin_shufflevector(l0_, h0_, 0, 1, 2, 3)); \
        dst[n][1] = __builtin_bit_cast(bf16x8, __builtin_shufflevector(l1_, h1_, 0, 1, 2, 3)); } } while (0)
#define G_LDB(dst, b, h) do { if ((b) == 0 && (h) == 0) G_LDB_(dst, 0, 8192); else if ((b) == 0) G_LDB_(dst, 16384, 24576); \
        else if ((h) == 0) G_LDB_(dst, 32768, 40960); else G_LDB_(dst, 49152, 57344); } while (0)
#define G_WAIT_LB(F) asm volatile("s_waitcnt lgkmcnt(0)" : "+v"(F[0][0]), "+v"(F[0][1]), "+v"(F[1][0]), "+v"(F[1][1]) :: "memory")
#define G_MMA(ai, bj, At, Bt_) do { __builtin_amdgcn_s_setprio(1); _Pragma("unroll") for (int m = 0; m < 4; ++m) _Pragma("unroll") for (int n = 0; n < 2; ++n) _Pragma("unroll") for (int k = 0; k < 2; ++k) \
        acc[ai][bj][m][n] = __builtin_amdgcn_mfma_f32_16x16x32_bf16(Bt_[n][k], At[m][k], acc[ai][bj][m][n], 0, 0, 0); __builtin_amdgcn_s_setprio(0); } while (0)
#define G_WAIT_V(n) asm volatile("s_waitcnt vmcnt(" #n ")" ::: "memory")
#define G_WAIT_L(n) asm volatile("s_waitcnt lgkmcnt(" #n ")" ::: "memory")
#define G_BAR __builtin_amdgcn_s_barrier()
#define G_SCHED __builtin_amdgcn_sched_barrier(0)
    f32x4 acc[2][2][4][2];
#pragma unroll
    for (int a = 0; a < 2; ++a)
#pragma unroll
        for (int bb = 0; bb < 2; ++bb)
#pragma unroll
            for (int m = 0; m < 4; ++m)
#pragma unroll
                for (int n = 0; n < 2; ++n) acc[a][bb][m][n] = (f32x4){0.f, 0.f, 0.f, 0.f};
    bf16x8 At[4][2], B0[2][2], B1[2][2];
    const char* cA = baseA + (size_t)pml * BM * rstride;
    const char* cB = (const char*)p.w + (size_t)pn * BM * 2;
    STAMP(myws, b, 0);
    if (remote) wait_chunk(flags, call, tmo);
    STAMP(myws, b, 1);
    G_STAGE(G_SB(0, 0), cB, voffB); G_STAGE(G_SA(0, 0), cA, voffA); G_STAGE(G_SB(0, 1), cB + hstepB, voffB); G_STAGE(G_SA(0, 1), cA + hstepA, voffA);
    if (wr == 1) G_BAR;
    G_WAIT_V(4); G_BAR;
    G_STAGE(G_SB(1, 0), cB + kstepB, voffB); G_STAGE(G_SA(1, 0), cA + kstep, voffA); G_STAGE(G_SB(1, 1), cB + hstepB + kstepB, voffB);
    G_WAIT_V(6); G_BAR;
    for (int t = 0; t < NT; t += 2) {
        const bool last = (t == NT - 2);
        const char* a0 = cA + (size_t)(t / TPC) * cstride + (size_t)(t % TPC) * kstep;
        const char* a1 = a0 + kstep;
        const char* a2 = last ? cA : cA + (size_t)((t + 2) / TPC) * cstride + (size_t)((t + 2) % TPC) * kstep;
        const char* b2 = last ? cB : cB + (size_t)(t + 2) * kstepB;
        const char* a3 = a2 + kstep; const char* b3 = b2 + kstepB;
        if (remote && !last && ((t + 2) % TPC) == 0) { wait_chunk(flags + ((t + 2) / TPC) * NCOPY, call, tmo); STAMP(myws, b, 1 + (t + 2) / TPC); }
        G_LDB(B0, 0, 0); G_SCHED; G_LDA(At, 0, 0); G_STAGE(G_SA(1, 1), a1 + hstepA, voffA);
        G_WAIT_L(8); G_BAR; G_WAIT_LB(B0); G_MMA(0, 0, At, B0); G_BAR; G_SCHED;
        G_LDB(B1, 0, 1); G_STAGE(G_SB(0, 0), b2, voffB);
        G_BAR; G_WAIT_LB(B1); G_MMA(0, 1, At, B1); G_BAR;
        G_LDA(At, 0, 1); G_STAGE(G_SA(0, 0), a2, voffA);
        G_BAR; G_WAIT_L(0); G_MMA(1, 0, At, B0); G_BAR; G_SCHED;
        G_STAGE(G_SB(0, 1), b2 + hstepB, voffB);
        G_WAIT_V(6); G_BAR; G_MMA(1, 1, At, B1); G_BAR;
        G_LDB(B0, 1, 0); G_SCHED; G_LDA(At, 1, 0); G_STAGE(G_SA(0, 1), a2 + hstepA, voffA);
        G_WAIT_L(8); G_BAR; G_WAIT_LB(B0); G_MMA(0, 0, At, B0); G_BAR; G_SCHED;
        G_LDB(B1, 1, 1); G_STAGE(G_SB(1, 0), b3, voffB);
        G_BAR; G_WAIT_LB(B1); G_MMA(0, 1, At, B1); G_BAR;
        G_LDA(At, 1, 1); G_STAGE(G_SA(1, 0), a3, voffA);
        G_BAR; G_WAIT_L(0); G_MMA(1, 0, At, B0); G_BAR; G_SCHED;
        G_STAGE(G_SB(1, 1), b3 + hstepB, voffB);
        G_WAIT_V(6); G_BAR; G_MMA(1, 1, At, B1); G_BAR;
    }
    STAMP(myws, b, 40);
    G_WAIT_V(0);
    if (wr == 0) G_BAR;
    G_BAR;
    {
        const bool bad = (__hip_atomic_load(tmo, __ATOMIC_RELAXED, __HIP_MEMORY_SCOPE_AGENT) == call);
#pragma unroll
        for (int ai = 0; ai < 2; ++ai)
#pragma unroll
            for (int m = 0; m < 4; ++m) {
                const int r = ai * HALF + wr * 64 + m * 16 + fr;
#pragma unroll
                for (int bj = 0; bj < 2; ++bj)
#pragma unroll
                    for (int n = 0; n < 2; ++n) {
                        const f32x4 v0 = acc[ai][bj][m][n];
                        u32x2 w; w.x = cvt_pk_bf16(v0[0], v0[1]); w.y = cvt_pk_bf16(v0[2], v0[3]);
                        if (bad) { w.x = 0x7fc07fc0u; w.y = 0x7fc07fc0u; }
                        const int ch = bj * 16 + wc * 4 + n * 2 + (fq >> 1);
                        *(LAS u32x2*)(lds + r * 512 + ((ch ^ (r & 31)) << 4) + (fq & 1) * 8) = w;
                    }
            }
        asm volatile("s_waitcnt lgkmcnt(0)" ::: "memory"); G_BAR; asm volatile("" ::: "memory");
        const int ch = tid & 31;
#pragma unroll
        for (int i = 0; i < 16; ++i) {
            const int r = (tid >> 5) + i * 16;
            const u32x4 w = *(const LAS u32x4*)(lds + r * 512 + ((ch ^ (r & 31)) << 4));
            *(u32x4*)(p.out + (size_t)(pm * BM + r) * NSH + pn * BM + ch * 8) = w;
        }
    }
    STAMP(myws, b, 41);
}

__global__ __launch_bounds__(512, 2) void ag_gemm_kernel(Params p) {
    extern __shared__ __attribute__((aligned(16))) unsigned char shm[];
    LAS unsigned char* lds = (LAS unsigned char*)shm;
    const unsigned call = derive_call((unsigned*)(p.ws[p.rank] + CTR), (LAS unsigned*)(lds + STAGE_BYTES));
    if (blockIdx.x < NCOPY) copy_phase(p, call);
    else gemm_phase(p, lds, call);
}

}

extern "C" void kernel_launch(int rank, int nranks, ncclComm_t comm,
                              hipStream_t stream,
                              void* const* d_in, const int* in_sizes, int n_in,
                              void* d_out, int out_size,
                              const int* in_dtypes, int out_dtype,
                              void* const* d_out_all,
                              void* const* d_ws_all, size_t ws_bytes) {
    static bool init = false;
    if (nranks != NR || rank < 0 || rank >= NR || ws_bytes < WS_NEED) { fprintf(stderr, "unsupported configuration\n"); return; }
    if (!init) {
        HIP_CHECK(hipFuncSetAttribute((const void*)ag_gemm_kernel, hipFuncAttributeMaxDynamicSharedMemorySize, LDS_BYTES));
        init = true;
    }
    Params p;
    p.x = (const bf16_t*)d_in[0];
    p.w = (const bf16_t*)d_in[1];
    p.out = (bf16_t*)d_out;
    for (int i = 0; i < NR; ++i) p.ws[i] = (char*)d_ws_all[i];
    p.rank = rank;
    ag_gemm_kernel<<<NCOPY + 128, 512, LDS_BYTES, stream>>>(p);
    HIP_CHECK(hipGetLastError());
}
```
